```python
import numpy as np

kernel_name = "gpu_gemm_rs_m4096_k4096_n32768_bf16_relu"


ABSMAX_RTOL = 0.008

COMPUTE_DTYPE = "bfloat16"
EPILOGUE = "relu"
IN_QUANT = None
OUT_QUANT = None
IN_DTYPES = {"x": "bfloat16", "w_mat": "bfloat16"}
WORLD_SIZE = 8

def _to_bf16(x):
    u = np.ascontiguousarray(x, dtype=np.float32).view(np.uint32)
    return ((u + 0x7FFF + ((u >> 16) & 1)) & 0xFFFF0000).view(np.float32)

def setup_inputs():

    x = _to_bf16(np.random.randn(4096, 4096).astype(np.float32))
    w_mat = _to_bf16((np.random.randn(4096, 32768) / np.sqrt(4096)).astype(np.float32))
    return {"x": x, "w_mat": w_mat}

def _epilogue(y):
    return np.maximum(y, 0.0).astype(np.float32)

def reference(x, w_mat):
    y = (x @ w_mat).astype(np.float32)
    return _epilogue(y)
```

```cpp
#include <hip/hip_runtime.h>
#include <rccl/rccl.h>
#include <cstdio>

#define HIP_CHECK(expr) do { hipError_t _e = (expr); if (_e != hipSuccess) { \
    fprintf(stderr, "HIP error %s at %s:%d\n", hipGetErrorString(_e), __FILE__, __LINE__); \
    return; } } while (0)

namespace {

#define LAS __attribute__((address_space(3)))
#define GAS __attribute__((address_space(1)))
typedef unsigned short bf16_t;
typedef short bf16x8 __attribute__((ext_vector_type(8)));
typedef short s16x4 __attribute__((ext_vector_type(4)));
typedef float f32x4 __attribute__((ext_vector_type(4)));
typedef unsigned u32x4 __attribute__((ext_vector_type(4)));
typedef GAS u32x4 g128;

constexpr int NR = 8;
constexpr int MTOT = 4096, KTOT = 4096, NTOT = 32768;
constexpr int KS = KTOT / NR;
constexpr int NS = NTOT / NR;
constexpr int MS = MTOT / NR;
constexpr int KC = 128;
constexpr int JC = KS / KC;
constexpr int NCH = KTOT / KC;
constexpr int NC = 4;
constexpr int NG = 28;
constexpr int NBLK = 256;

constexpr size_t CTL_BYTES = 256 * 1024;
constexpr size_t CHB = (size_t)4096 * KC * 2;
constexpr size_t STG = (size_t)NCH * CHB;
constexpr size_t OFF_A = CTL_BYTES;
constexpr size_t OFF_B = OFF_A + 2 * STG;
constexpr size_t OFF_O = OFF_B + 2 * STG;
constexpr size_t WS_NEED = OFF_O + (size_t)MTOT * NS * 2;
constexpr int CW = 64;
constexpr int C_TICKET = 0;
constexpr int C_TMO = 1;
constexpr int C_XCNT = 2;
constexpr int C_WCNT = C_XCNT + NR * JC;
constexpr int C_ODONE = C_WCNT + NR * 8;
constexpr int C_TILE = C_ODONE + NR;
constexpr int C_END = C_TILE + 256;
static_assert((size_t)C_END * CW * 4 <= CTL_BYTES, "ctl");

constexpr unsigned long long DEADLINE_TICKS = 50000000ull;

constexpr int BK = 64, HALF = 128, HTB = HALF * BK * 2, STAGE_BYTES = 8 * HTB;
constexpr int LDS_BYTES = STAGE_BYTES + 1024;

__device__ __forceinline__ int lds_byte(int r, int c) { const int st = (r >> 4) * 2 + (c >> 5), rr = r & 15, cc = c & 31, ob = rr * 64 + cc * 2; return st * 1024 + (ob ^ (((ob >> 9) & 1) << 5)); }
__device__ __forceinline__ void stage_rc(int b, int& R, int& C) { const int st = b / 1024, sb = b % 1024, swz = sb ^ (((sb >> 9) & 1) << 5); R = (st >> 1) * 16 + swz / 64; C = (st & 1) * 32 + (swz % 64) / 2; }
__device__ __forceinline__ int perm32(int rho) { const int n = rho >> 4, i = rho & 15; return 8 * (i >> 2) + 4 * n + (i & 3); }
__device__ __forceinline__ unsigned cvt_pk_bf16(float lo, float hi) { unsigned r; asm volatile("v_cvt_pk_bf16_f32 %0, %1, %2" : "=v"(r) : "v"(lo), "v"(hi)); return r; }
__device__ __forceinline__ u32x4 pack8(f32x4 a, f32x4 b) {
    const f32x4 z = {0.f, 0.f, 0.f, 0.f};
    a = __builtin_elementwise_max(a, z); b = __builtin_elementwise_max(b, z);
    u32x4 w; w.x = cvt_pk_bf16(a[0], a[1]); w.y = cvt_pk_bf16(a[2], a[3]); w.z = cvt_pk_bf16(b[0], b[1]); w.w = cvt_pk_bf16(b[2], b[3]);
    return w;
}

struct Params {
    const bf16_t* x;
    const bf16_t* w;
    bf16_t* out;
    void* const* out_all;
    void* const* ws_all;
    int rank;
};

__device__ __forceinline__ unsigned* ctl(void* ws, int line) { return (unsigned*)ws + (size_t)line * CW; }
#define STAMP(ws, k) do {} while (0)

template <int SLEEP>
__device__ __forceinline__ void spin_ge(unsigned* flag, unsigned need, unsigned* tmo, unsigned call) {
    const unsigned long long t0 = __builtin_amdgcn_s_memrealtime();
    while (__hip_atomic_load(flag, __ATOMIC_RELAXED, __HIP_MEMORY_SCOPE_SYSTEM) < need) {
        if (__hip_atomic_load(tmo, __ATOMIC_RELAXED, __HIP_MEMORY_SCOPE_AGENT) == call
            || __builtin_amdgcn_s_memrealtime() - t0 > DEADLINE_TICKS) {
            __hip_atomic_store(tmo, call, __ATOMIC_RELAXED, __HIP_MEMORY_SCOPE_AGENT);
            break;
        }
        __builtin_amdgcn_s_sleep(SLEEP);
    }
}

__device__ __forceinline__ __amdgpu_buffer_rsrc_t mk_rsrc(const void* base, unsigned bytes) {
    const unsigned long long a = (unsigned long long)base;
    const unsigned lo = __builtin_amdgcn_readfirstlane((unsigned)a), hi = __builtin_amdgcn_readfirstlane((unsigned)(a >> 32));
    return __builtin_amdgcn_make_buffer_rsrc((void*)(((unsigned long long)hi << 32) | lo), (short)0, (int)bytes, 0x00020000);
}
__device__ __forceinline__ void st_sys(__amdgpu_buffer_rsrc_t rs, unsigned off, u32x4 v) { __builtin_amdgcn_raw_buffer_store_b128(v, rs, (int)off, 0, 17); }

constexpr int NDW = 4;
constexpr int NW = NC * NDW;
constexpr int NGRP = 12;
#define SCHED_ID 1
__device__ constexpr int SCHED_[NGRP] = {0, 4, 1, 5, 2, 6, 3, 7, 8, 9, 10, 11};
constexpr int TP = 32;
constexpr int TW_BYTES = 128 * TP * 2;

__device__ __forceinline__ void lds_inc(LAS unsigned* w) { __hip_atomic_fetch_add(w, 1u, __ATOMIC_RELAXED, __HIP_MEMORY_SCOPE_WORKGROUP); }
__device__ __forceinline__ unsigned lds_get(LAS unsigned* w) { return __hip_atomic_load(w, __ATOMIC_RELAXED, __HIP_MEMORY_SCOPE_WORKGROUP); }

template <int S, class LD, class ST, class SG>
__device__ __forceinline__ void stream_pieces(int first, int np, LD ld, ST st, SG sig) {
    u32x4 a[8], b[8];
    int k = first;
    ld(k, a);
    ld(k + S, b); st(k, a);
    for (; k + 3 * S < np; k += 2 * S) {
        ld(k + 2 * S, a); st(k + S, b);
        asm volatile("s_waitcnt vmcnt(16)" ::: "memory"); sig(k);
        ld(k + 3 * S, b); st(k + 2 * S, a);
        asm volatile("s_waitcnt vmcnt(16)" ::: "memory"); sig(k + S);
    }
    st(k + S, b);
    asm volatile("s_waitcnt vmcnt(8)" ::: "memory"); sig(k);
    asm volatile("s_waitcnt vmcnt(0)" ::: "memory"); sig(k + S);
}

__device__ void comm_role(const Params& p, LAS unsigned char* lds, unsigned call, int d, int ci) {
    const int tid = threadIdx.x, wid = __builtin_amdgcn_readfirstlane(tid >> 6), lane = tid & 63;
    const int rank = p.rank;
    char* myws = (char*)p.ws_all[rank];
    char* dws = (char*)p.ws_all[d];
    const unsigned par = call & 1u;
    unsigned* tmo = ctl(myws, C_TMO);
    LAS unsigned* cnt = (LAS unsigned*)(lds + NDW * TW_BYTES);
    if (tid < 64) cnt[tid] = 0u;
    __syncthreads();
    const bool remote = d != rank;
    if (wid > NDW) return;

    if (wid == NDW) {
        if (lane == 0) {
            for (int g = 0; g < NGRP; ++g) {
                const int code = SCHED_[g];
                while (lds_get(cnt + g) < (code < 4 ? 32u : 16u)) __builtin_amdgcn_s_sleep(1);
                unsigned* ctr = code < 4 ? ctl(dws, C_XCNT + rank * JC + code) : ctl(dws, C_WCNT + rank * 8 + (code - 4));
                __hip_atomic_fetch_add(ctr, 1u, __ATOMIC_RELAXED, __HIP_MEMORY_SCOPE_SYSTEM);
                if (code == 7) STAMP(myws, 2);
            }
            STAMP(myws, 3);
            if (remote) {
                for (int tt = 0; tt < 8; ++tt) {
                    const int t = tt * NC + ci;
                    spin_ge<16>(ctl(myws, C_TILE + (2 * d + (t & 1)) * 16 + (t >> 1)), 8u * call, tmo, call);
                    __hip_atomic_store(cnt + 25, (unsigned)(tt + 1), __ATOMIC_RELAXED, __HIP_MEMORY_SCOPE_WORKGROUP);
                    if (tt == 0) STAMP(myws, 4);
                    if (tt == 6) STAMP(myws, 5);
                    if (tt == 7) STAMP(myws, 6);
                }
                while (lds_get(cnt + 24) < 128u) __builtin_amdgcn_s_sleep(1);
                __hip_atomic_fetch_add(ctl(dws, C_ODONE + rank), 8u, __ATOMIC_RELAXED, __HIP_MEMORY_SCOPE_SYSTEM);
                STAMP(myws, 7);
            } else if (ci == 0) {
                for (int s = 0; s < NR; ++s) if (s != rank) spin_ge<2>(ctl(myws, C_ODONE + s), 32u * call, tmo, call);
                __builtin_amdgcn_fence(__ATOMIC_ACQUIRE, "");
                if (__hip_atomic_load(tmo, __ATOMIC_RELAXED, __HIP_MEMORY_SCOPE_AGENT) == call) p.out[(size_t)rank * NS] = (bf16_t)0x7fc0u;
                STAMP(myws, 8);
            }
        }
        return;
    }

    LAS bf16_t* T = (LAS bf16_t*)(lds + wid * TW_BYTES);
    const int gw = ci * NDW + wid;
    if (wid == 0) STAMP(myws, 1);
    auto push_x = [&](int j, int g) {
        const g128* src = (const g128*)(p.x + j * KC) + (lane >> 4) * (KS / 8) + (lane & 15);
        const __amdgpu_buffer_rsrc_t dst = mk_rsrc(dws + OFF_A + par * STG + (size_t)(rank * JC + j) * CHB, (unsigned)CHB);
        stream_pieces<NW>(gw, 128,
            [&](int P, u32x4 (&v)[8]) {
#pragma unroll
                for (int u = 0; u < 8; ++u) v[u] = src[(size_t)(P * 32 + u * 4) * (KS / 8)]; },
            [&](int P, u32x4 (&v)[8]) {
#pragma unroll
                for (int u = 0; u < 8; ++u) st_sys(dst, (unsigned)(P * 512 + u * 64 + lane) * 16u, v[u]); },
            [&](int P) { if (lane == 0) lds_inc(cnt + g); });
    };
    auto push_w = [&](int part, int j, int g) {
        const g128* src = (const g128*)(p.w + (size_t)(j * KC) * NTOT + (size_t)d * NS) + (size_t)(lane >> 2) * (NTOT / 8) + (lane & 3);
        const __amdgpu_buffer_rsrc_t dst = mk_rsrc(dws + OFF_B + par * STG + (size_t)(rank * JC + j) * CHB, (unsigned)CHB);
        stream_pieces<NW>(part * 64 + gw, part * 64 + 64,
            [&](int P, u32x4 (&v)[8]) {
#pragma unroll
                for (int u = 0; u < 8; ++u) v[u] = src[(size_t)(u * 16) * (NTOT / 8) + P * 4]; },
            [&](int P, u32x4 (&v)[8]) {
#pragma unroll
                for (int u = 0; u < 8; ++u) *(LAS u32x4*)(T + (u * 16 + (lane >> 2)) * TP + (lane & 3) * 8) = v[u];
#pragma unroll
                for (int u = 0; u < 8; ++u) {
                    const int kb = (u & 3) * 4 + (lane >> 4), q = (lane & 15) >> 2, pp = lane & 3;
                    LAS bf16_t* a = T + (kb * 8 + q) * TP + (u >> 2) * 16 + pp * 4;
                    const s16x4 lo = __builtin_amdgcn_ds_read_tr16_b64_v4i16((LAS s16x4*)a);
                    const s16x4 hi = __builtin_amdgcn_ds_read_tr16_b64_v4i16((LAS s16x4*)(a + 4 * TP));
                    const bf16x8 o = __builtin_shufflevector(lo, hi, 0, 1, 2, 3, 4, 5, 6, 7);
                    st_sys(dst, (unsigned)(P * 512 + u * 64 + lane) * 16u, __builtin_bit_cast(u32x4, o));
                } },
            [&](int P) { if (lane == 0) lds_inc(cnt + g); });
    };
    for (int g = 0; g < NGRP; ++g) {
        const int code = SCHED_[g];
        if (code < 4) push_x(code, g); else push_w((code - 4) >> 2, (code - 4) & 3, g);
    }
    if (remote) {
        const g128* src = (const g128*)(myws + OFF_O) + (size_t)(2 * d * 256 + (lane >> 5)) * (NS / 8) + (lane & 31);
        const __amdgpu_buffer_rsrc_t dst = mk_rsrc(p.out_all[d], MS * NTOT * 2);
        const unsigned doff = (unsigned)((lane >> 5) * (NTOT / 8) + rank * (NS / 8) + (lane & 31));
        stream_pieces<NDW>(wid, 128,
            [&](int P, u32x4 (&v)[8]) {
                const int t = (P >> 4) * NC + ci, row = (t & 1) * 256 + (P & 15) * 16, col = (t >> 1) * 32;
                while (lds_get(cnt + 25) <= (unsigned)(P >> 4)) __builtin_amdgcn_s_sleep(8);
                asm volatile("" ::: "memory");
#pragma unroll
                for (int u = 0; u < 8; ++u) v[u] = src[(size_t)(row + u * 2) * (NS / 8) + col]; },
            [&](int P, u32x4 (&v)[8]) {
                const int t = (P >> 4) * NC + ci, row = (t & 1) * 256 + (P & 15) * 16, col = (t >> 1) * 32;
#pragma unroll
                for (int u = 0; u < 8; ++u) st_sys(dst, (doff + (unsigned)((row + u * 2) * (NTOT / 8) + col)) * 16u, v[u]); },
            [&](int P) { if (lane == 0) lds_inc(cnt + 24); });
    }
}

__device__ void gemm_role(const Params& p, LAS unsigned char* lds, unsigned call, int xg, int idx) {
    const int tid = threadIdx.x, wid = __builtin_amdgcn_readfirstlane(tid >> 6), lane = tid & 63, wr = wid >> 2, wc = wid & 3, fr = lane & 15, fq = lane >> 4;
    const int rank = p.rank;
    char* myws = (char*)p.ws_all[rank];
    unsigned* tmo = ctl(myws, C_TMO);
    const unsigned par = call & 1u;
    const int pm = (xg & 3) * 4 + (idx & 3), pn0 = idx < 16 ? (xg >> 2) * 4 + (idx >> 2) : 8 + (xg >> 2) * 3 + ((idx - 16) >> 2), pn1 = 14 + (xg >> 2);
    const int nun = idx < 4 ? 2 : 1;

    unsigned voffA[2], voffB[2];
#pragma unroll
    for (int i = 0; i < 2; ++i) { int R, C; stage_rc(tid * 16 + i * 8192, R, C); const int Rb = (R & ~31) + perm32(R & 31);
        voffA[i] = (unsigned)(R * KC + C) * 2u; voffB[i] = (unsigned)(((Rb >> 4) * 16 + (C >> 3)) * 256 + (Rb & 15) * 16); }
    constexpr size_t kstep = BK * 2;
    constexpr size_t kstepB = 8 * 256;
    constexpr size_t hstep = (size_t)HALF * KC * 2;
    constexpr size_t tstep = 2 * hstep;
    const unsigned ldsw = (unsigned)wid * 1024u;
    const int aoff = lds_byte(wr * 64 + fr, fq * 8), boff = lds_byte(wc * 32 + fr, fq * 8);
#define SA(b, h) (((b) * 2 + (h)) * HTB)
#define SB(b, h) ((4 + (b) * 2 + (h)) * HTB)
#define STAGE(bufoff, gbase, voff) do { _Pragma("unroll") for (int _i = 0; _i < 2; ++_i) \
        __builtin_amdgcn_global_load_lds((const unsigned*)((const char*)(gbase) + (voff)[_i]), (LAS unsigned*)(lds + (bufoff) + ldsw + _i * 8192), 16, 0, 0); } while (0)
#define LDA(dst, b, h) do { _Pragma("unroll") for (int m = 0; m < 4; ++m) _Pragma("unroll") for (int k = 0; k < 2; ++k) dst[m][k] = *(const LAS bf16x8*)(lds + SA(b, h) + aoff + m * 2048 + k * 1024); } while (0)
#define LDB(dst, b, h) do { _Pragma("unroll") for (int n = 0; n < 2; ++n) _Pragma("unroll") for (int k = 0; k < 2; ++k) dst[n][k] = *(const LAS bf16x8*)(lds + SB(b, h) + boff + n * 2048 + k * 1024); } while (0)
#define MMA(ai, bj, At, Bt) do { __builtin_amdgcn_s_setprio(1); _Pragma("unroll") for (int m = 0; m < 4; ++m) _Pragma("unroll") for (int n = 0; n < 2; ++n) _Pragma("unroll") for (int k = 0; k < 2; ++k) \
        acc[ai][bj][m][n] = __builtin_amdgcn_mfma_f32_16x16x32_bf16(Bt[n][k], At[m][k], acc[ai][bj][m][n], 0, 0, 0); __builtin_amdgcn_s_setprio(0); } while (0)
#define WAIT_V(n) asm volatile("s_waitcnt vmcnt(" #n ")" ::: "memory")
#define WAIT_L(n) asm volatile("s_waitcnt lgkmcnt(" #n ")" ::: "memory")
#define BAR __builtin_amdgcn_s_barrier()
#define SCHED __builtin_amdgcn_sched_barrier(0)

    const char* Abase = myws + OFF_A + par * STG + (size_t)pm * tstep;
    const char* Bbase = myws + OFF_B + par * STG;
    auto chunk_of = [&](int q) { const int j = q >> 3, s = q & 7; return ((rank + s) & 7) * JC + j; };

    int known = 0;
    auto ready = [&](int h, int q) {
        if (wid == 0 && known <= q) {
            const unsigned long long t0 = __builtin_amdgcn_s_memrealtime();
            const int kc = chunk_of(lane & 31);
            unsigned* fx = ctl(myws, C_XCNT + kc);
            unsigned* fw = ctl(myws, C_WCNT + (kc >> 2) * 8 + ((h ? pn1 : pn0) >> 3) * 4 + (kc & 3));
            const unsigned need = (unsigned)NC * call;
            for (;;) {
                const unsigned vx = __hip_atomic_load(fx, __ATOMIC_RELAXED, __HIP_MEMORY_SCOPE_SYSTEM);
                const unsigned vw = __hip_atomic_load(fw, __ATOMIC_RELAXED, __HIP_MEMORY_SCOPE_SYSTEM);
                const unsigned long long ok = __ballot(vx >= need && vw >= need) | 0xffffffff00000000ull;
                known = (~ok == 0ull) ? 32 : __builtin_ctzll(~ok);
                if (known > q) break;
                if (__builtin_amdgcn_readfirstlane(__hip_atomic_load(tmo, __ATOMIC_RELAXED, __HIP_MEMORY_SCOPE_AGENT)) == call
                    || __builtin_amdgcn_s_memrealtime() - t0 > DEADLINE_TICKS) {
                    if (lane == 0) __hip_atomic_store(tmo, call, __ATOMIC_RELAXED, __HIP_MEMORY_SCOPE_AGENT);
                    known = 32; break;
                }
                __builtin_amdgcn_s_sleep(4);
            }
        }
        asm volatile("" ::: "memory"); BAR; asm volatile("" ::: "memory");
    };

    f32x4 acc[2][2][4][2];
#pragma unroll
    for (int a = 0; a < 2; ++a)
#pragma unroll
        for (int b = 0; b < 2; ++b)
#pragma unroll
            for (int m = 0; m < 4; ++m)
#pragma unroll
                for (int n = 0; n < 2; ++n) acc[a][b][m][n] = (f32x4){0.f, 0.f, 0.f, 0.f};
    bf16x8 At[4][2], B0[2][2], B1[2][2];

    {
        const char* cA = Abase + (size_t)chunk_of(0) * CHB;
        const char* cB = Bbase + (size_t)chunk_of(0) * CHB + (size_t)pn0 * tstep;
        STAMP(myws, 1);
        ready(0, 0);
        STAMP(myws, 2);
        STAGE(SB(0, 0), cB, voffB); STAGE(SA(0, 0), cA, voffA); STAGE(SB(0, 1), cB + hstep, voffB); STAGE(SA(0, 1), cA + hstep, voffA);
        if (wr == 1) BAR;
        WAIT_V(4); BAR;
        STAGE(SB(1, 0), cB + kstepB, voffB); STAGE(SA(1, 0), cA + kstep, voffA); STAGE(SB(1, 1), cB + hstep + kstepB, voffB);
        WAIT_V(6); BAR;
    }
    for (int h = 0; h < nun; ++h) {
        const int pn = h ? pn1 : pn0;
        for (int q = 0; q < NCH; ++q) {
            const bool last = (q == NCH - 1);
            const bool nxt = last && h + 1 < nun;
            const int hn = nxt ? h + 1 : h, qn = last ? 0 : q + 1;
            const char* a1 = Abase + (size_t)chunk_of(q) * CHB + kstep;
            const char* a2 = Abase + (size_t)chunk_of(qn) * CHB;
            const char* b2 = Bbase + (size_t)chunk_of(qn) * CHB + (size_t)(hn ? pn1 : pn0) * tstep;
            const char* a3 = a2 + kstep; const char* b3 = b2 + kstepB;
            if (nxt) known = 0;
            if (wid == 0) STAMP(myws, 16 + (h * 32 + q) * 2);
            ready(hn, qn);
            if (wid == 0) STAMP(myws, 17 + (h * 32 + q) * 2);
            LDB(B0, 0, 0); SCHED; LDA(At, 0, 0); STAGE(SA(1, 1), a1 + hstep, voffA);
            WAIT_L(8); BAR; WAIT_L(0); MMA(0, 0, At, B0); BAR; SCHED;
            LDB(B1, 0, 1); STAGE(SB(0, 0), b2, voffB);
            BAR; WAIT_L(0); MMA(0, 1, At, B1); BAR;
            LDA(At, 0, 1); STAGE(SA(0, 0), a2, voffA);
            BAR; WAIT_L(0); MMA(1, 0, At, B0); BAR; SCHED;
            STAGE(SB(0, 1), b2 + hstep, voffB);
            WAIT_V(6); BAR; MMA(1, 1, At, B1); BAR;
            LDB(B0, 1, 0); SCHED; LDA(At, 1, 0); STAGE(SA(0, 1), a2 + hstep, voffA);
            WAIT_L(8); BAR; WAIT_L(0); MMA(0, 0, At, B0); BAR; SCHED;
            LDB(B1, 1, 1); STAGE(SB(1, 0), b3, voffB);
            BAR; WAIT_L(0); MMA(0, 1, At, B1); BAR;
            LDA(At, 1, 1); STAGE(SA(1, 0), a3, voffA);
            BAR; WAIT_L(0); MMA(1, 0, At, B0); BAR; SCHED;
            STAGE(SB(1, 1), b3 + hstep, voffB);
            WAIT_V(6); BAR; MMA(1, 1, At, B1); BAR;
        }
        STAMP(myws, 3 + 2 * h);
        {
            const bool own = (pm >> 1) == rank;
            const int row0 = wr * 64 + fr, col0 = wc * 32 + 8 * fq;
            if (own) {
                bf16_t* base = p.out + (size_t)((pm & 1) * 256) * NTOT + (size_t)rank * NS + pn * 256;
#pragma unroll
                for (int ai = 0; ai < 2; ++ai)
#pragma unroll
                    for (int m = 0; m < 4; ++m) { bf16_t* rowp = base + (size_t)(row0 + ai * HALF + m * 16) * NTOT + col0;
#pragma unroll
                        for (int bj = 0; bj < 2; ++bj) *(u32x4*)(rowp + bj * HALF) = pack8(acc[ai][bj][m][0], acc[ai][bj][m][1]); }
            } else {
                const __amdgpu_buffer_rsrc_t rs = mk_rsrc(myws + OFF_O + ((size_t)(pm * 256) * NS + pn * 256) * 2, 256 * NS * 2);
#pragma unroll
                for (int ai = 0; ai < 2; ++ai)
#pragma unroll
                    for (int m = 0; m < 4; ++m) { const int ro = ((row0 + ai * HALF + m * 16) * NS + col0) * 2;
#pragma unroll
                        for (int bj = 0; bj < 2; ++bj) __builtin_amdgcn_raw_buffer_store_b128(pack8(acc[ai][bj][m][0], acc[ai][bj][m][1]), rs, ro + bj * HALF * 2, 0, 16); }
                asm volatile("s_waitcnt vmcnt(0)" ::: "memory");
                if (lane == 0) __hip_atomic_fetch_add(ctl(myws, C_TILE + pm * 16 + pn), 1u, __ATOMIC_RELAXED, __HIP_MEMORY_SCOPE_AGENT);
            }
        }
        STAMP(myws, 4 + 2 * h);
#pragma unroll
        for (int a = 0; a < 2; ++a)
#pragma unroll
            for (int b = 0; b < 2; ++b)
#pragma unroll
                for (int m = 0; m < 4; ++m)
#pragma unroll
                    for (int n = 0; n < 2; ++n) acc[a][b][m][n] = (f32x4){0.f, 0.f, 0.f, 0.f};
    }
    WAIT_V(0);
    if (wr == 0) BAR;
    BAR;
#undef SA
#undef SB
#undef STAGE
#undef LDA
#undef LDB
#undef MMA
#undef WAIT_V
#undef WAIT_L
#undef BAR
#undef SCHED
}

__global__ __launch_bounds__(512, 2) void fused_kernel(Params p) {
    extern __shared__ __attribute__((aligned(16))) unsigned char shm[];
    LAS unsigned char* lds = (LAS unsigned char*)shm;
    LAS unsigned* lw = (LAS unsigned*)(lds + STAGE_BYTES);
    const int grp = (int)blockIdx.x & 7, slot = (int)blockIdx.x >> 3;
    STAMP(p.ws_all[p.rank], 0);
    if (threadIdx.x == 0) {
        const unsigned t = __hip_atomic_fetch_add(ctl(p.ws_all[p.rank], C_TICKET), 1u, __ATOMIC_RELAXED, __HIP_MEMORY_SCOPE_AGENT);
        lw[0] = t / (unsigned)NBLK + 1u;
        __builtin_amdgcn_fence(__ATOMIC_ACQUIRE, "");
        asm volatile("s_waitcnt vmcnt(0)" ::: "memory");
    }
    __syncthreads();
    const unsigned call = __builtin_amdgcn_readfirstlane(lw[0]);
    __syncthreads();
    if (slot < NC) comm_role(p, lds, call, (p.rank + grp) & 7, slot);
    else gemm_role(p, lds, call, grp, slot - NC);
}

}

extern "C" void kernel_launch(int rank, int nranks, ncclComm_t comm,
                              hipStream_t stream,
                              void* const* d_in, const int* in_sizes, int n_in,
                              void* d_out, int out_size,
                              const int* in_dtypes, int out_dtype,
                              void* const* d_out_all,
                              void* const* d_ws_all, size_t ws_bytes) {
    if (nranks != NR || ws_bytes < WS_NEED || (unsigned)rank >= (unsigned)NR) { fprintf(stderr, "unexpected configuration\n"); return; }
    static bool attr[64] = {};
    int dev = 0; HIP_CHECK(hipGetDevice(&dev));
    if (!attr[dev & 63]) { HIP_CHECK(hipFuncSetAttribute((const void*)fused_kernel, hipFuncAttributeMaxDynamicSharedMemorySize, LDS_BYTES)); attr[dev & 63] = true; }
    Params p;
    p.x = (const bf16_t*)d_in[0]; p.w = (const bf16_t*)d_in[1]; p.out = (bf16_t*)d_out;
    p.out_all = d_out_all; p.ws_all = d_ws_all; p.rank = rank;
    hipLaunchKernelGGL(fused_kernel, dim3(NBLK), dim3(512), LDS_BYTES, stream, p);
    HIP_CHECK(hipGetLastError());
}
```

```cpp
#include <hip/hip_runtime.h>
#include <rccl/rccl.h>
#include <cstdio>

#define HIP_CHECK(expr) do { hipError_t _e = (expr); if (_e != hipSuccess) { \
    fprintf(stderr, "HIP error %s at %s:%d\n", hipGetErrorString(_e), __FILE__, __LINE__); \
    return; } } while (0)

namespace {

#define LAS __attribute__((address_space(3)))
#define GAS __attribute__((address_space(1)))
typedef unsigned short bf16_t;
typedef short bf16x8 __attribute__((ext_vector_type(8)));
typedef short s16x4 __attribute__((ext_vector_type(4)));
typedef float f32x4 __attribute__((ext_vector_type(4)));
typedef unsigned u32x4 __attribute__((ext_vector_type(4)));
typedef GAS u32x4 g128;

constexpr int NR = 8;
constexpr int MTOT = 4096, KTOT = 4096, NTOT = 32768;
constexpr int KS = KTOT / NR;
constexpr int NS = NTOT / NR;
constexpr int MS = MTOT / NR;
constexpr int KC = 128;
constexpr int JC = KS / KC;
constexpr int NCH = KTOT / KC;
constexpr int NC = 4;
constexpr int NG = 28;
constexpr int NBLK = 256;
#define CA_ 10
constexpr int CA = CA_;

constexpr size_t CTL_BYTES = 256 * 1024;
constexpr size_t CHB = (size_t)4096 * KC * 2;
constexpr size_t STG = (size_t)NCH * CHB;
constexpr size_t OFF_A = CTL_BYTES;
constexpr size_t OFF_B = OFF_A + 2 * STG;
constexpr size_t OFF_O = OFF_B + 2 * STG;
#define RS_ 2
constexpr int RS = RS_;
constexpr size_t OFF_P = OFF_O + (size_t)MTOT * NS * 2;
constexpr size_t WS_NEED = OFF_P + (size_t)32 * (RS - 1) * 256 * 256 * 4;
constexpr int CW = 64;
constexpr int C_TICKET = 0;
constexpr int C_TMO = 1;
constexpr int C_XCNT = 2;
constexpr int C_WCNT = C_XCNT + NR * JC;
constexpr int C_WCOL = C_WCNT + NR * 8;
constexpr int C_ODONE = C_WCOL + NR * 16;
constexpr int C_TILE = C_ODONE + NR;
constexpr int C_PART = C_TILE + 256;
constexpr int C_END = C_PART + 32 * 3;
static_assert((size_t)C_END * CW * 4 <= CTL_BYTES, "ctl");

constexpr unsigned long long DEADLINE_TICKS = 50000000ull;

constexpr int BK = 64, HALF = 128, HTB = HALF * BK * 2, STAGE_BYTES = 8 * HTB;
constexpr int LDS_BYTES = STAGE_BYTES + 1024;

__device__ __forceinline__ int lds_byte(int r, int c) { const int st = (r >> 4) * 2 + (c >> 5), rr = r & 15, cc = c & 31, ob = rr * 64 + cc * 2; return st * 1024 + (ob ^ (((ob >> 9) & 1) << 5)); }
__device__ __forceinline__ void stage_rc(int b, int& R, int& C) { const int st = b / 1024, sb = b % 1024, swz = sb ^ (((sb >> 9) & 1) << 5); R = (st >> 1) * 16 + swz / 64; C = (st & 1) * 32 + (swz % 64) / 2; }
__device__ __forceinline__ int perm32(int rho) { const int n = rho >> 4, i = rho & 15; return 8 * (i >> 2) + 4 * n + (i & 3); }
__device__ __forceinline__ unsigned cvt_pk_bf16(float lo, float hi) { unsigned r; asm volatile("v_cvt_pk_bf16_f32 %0, %1, %2" : "=v"(r) : "v"(lo), "v"(hi)); return r; }
__device__ __forceinline__ u32x4 pack8(f32x4 a, f32x4 b) {
    const f32x4 z = {0.f, 0.f, 0.f, 0.f};
    a = __builtin_elementwise_max(a, z); b = __builtin_elementwise_max(b, z);
    u32x4 w; w.x = cvt_pk_bf16(a[0], a[1]); w.y = cvt_pk_bf16(a[2], a[3]); w.z = cvt_pk_bf16(b[0], b[1]); w.w = cvt_pk_bf16(b[2], b[3]);
    return w;
}

struct Params {
    const bf16_t* x;
    const bf16_t* w;
    bf16_t* out;
    void* const* out_all;
    void* const* ws_all;
    int rank;
};

__device__ __forceinline__ unsigned* ctl(void* ws, int line) { return (unsigned*)ws + (size_t)line * CW; }
#define STAMP(ws, k) do {} while (0)

template <int SLEEP>
__device__ __forceinline__ void spin_ge(unsigned* flag, unsigned need, unsigned* tmo, unsigned call) {
    const unsigned long long t0 = __builtin_amdgcn_s_memrealtime();
    while (__hip_atomic_load(flag, __ATOMIC_RELAXED, __HIP_MEMORY_SCOPE_SYSTEM) < need) {
        if (__hip_atomic_load(tmo, __ATOMIC_RELAXED, __HIP_MEMORY_SCOPE_AGENT) == call
            || __builtin_amdgcn_s_memrealtime() - t0 > DEADLINE_TICKS) {
            __hip_atomic_store(tmo, call, __ATOMIC_RELAXED, __HIP_MEMORY_SCOPE_AGENT);
            break;
        }
        __builtin_amdgcn_s_sleep(SLEEP);
    }
}

__device__ __forceinline__ __amdgpu_buffer_rsrc_t mk_rsrc(const void* base, unsigned bytes) {
    const unsigned long long a = (unsigned long long)base;
    const unsigned lo = __builtin_amdgcn_readfirstlane((unsigned)a), hi = __builtin_amdgcn_readfirstlane((unsigned)(a >> 32));
    return __builtin_amdgcn_make_buffer_rsrc((void*)(((unsigned long long)hi << 32) | lo), (short)0, (int)bytes, 0x00020000);
}
__device__ __forceinline__ void st_sys(__amdgpu_buffer_rsrc_t rs, unsigned off, u32x4 v) { __builtin_amdgcn_raw_buffer_store_b128(v, rs, (int)off, 0, 17); }

constexpr int NDW = 4;
constexpr int NW = NC * NDW;
constexpr int NGRP = 12;
#define SCHED_ID 3
__device__ constexpr int SCHED_[NGRP] = {0, 4, 1, 5, 2, 6, 8, 3, 7, 9, 10, 11};
constexpr int TP = 32;
constexpr int TW_BYTES = 128 * TP * 2;

__device__ __forceinline__ void lds_inc(LAS unsigned* w) { __hip_atomic_fetch_add(w, 1u, __ATOMIC_RELAXED, __HIP_MEMORY_SCOPE_WORKGROUP); }
__device__ __forceinline__ unsigned lds_get(LAS unsigned* w) { return __hip_atomic_load(w, __ATOMIC_RELAXED, __HIP_MEMORY_SCOPE_WORKGROUP); }

template <int S, class LD, class ST, class SG>
__device__ __forceinline__ void stream_pieces(int first, int np, LD ld, ST st, SG sig) {
    u32x4 a[8], b[8];
    int k = first;
    ld(k, a);
    if (k + S >= np) { st(k, a); asm volatile("s_waitcnt vmcnt(0)" ::: "memory"); sig(k); return; }
    ld(k + S, b); st(k, a);
    for (; k + 3 * S < np; k += 2 * S) {
        ld(k + 2 * S, a); st(k + S, b);
        asm volatile("s_waitcnt vmcnt(16)" ::: "memory"); sig(k);
        ld(k + 3 * S, b); st(k + 2 * S, a);
        asm volatile("s_waitcnt vmcnt(16)" ::: "memory"); sig(k + S);
    }
    if (k + 2 * S < np) {
        ld(k + 2 * S, a); st(k + S, b);
        asm volatile("s_waitcnt vmcnt(16)" ::: "memory"); sig(k);
        st(k + 2 * S, a);
        asm volatile("s_waitcnt vmcnt(8)" ::: "memory"); sig(k + S);
        asm volatile("s_waitcnt vmcnt(0)" ::: "memory"); sig(k + 2 * S);
    } else {
        st(k + S, b);
        asm volatile("s_waitcnt vmcnt(8)" ::: "memory"); sig(k);
        asm volatile("s_waitcnt vmcnt(0)" ::: "memory"); sig(k + S);
    }
}

__device__ void comm_role(const Params& p, LAS unsigned char* lds, unsigned call, int d, int ci) {
    const int tid = threadIdx.x, wid = __builtin_amdgcn_readfirstlane(tid >> 6), lane = tid & 63;
    const int rank = p.rank;
    char* myws = (char*)p.ws_all[rank];
    char* dws = (char*)p.ws_all[d];
    const unsigned par = call & 1u;
    unsigned* tmo = ctl(myws, C_TMO);
    LAS unsigned* cnt = (LAS unsigned*)(lds + NDW * TW_BYTES);
    if (tid < 64) cnt[tid] = 0u;
    __syncthreads();
    const bool remote = d != rank;
    if (wid > NDW) return;

    if (wid == NDW) {
        if (lane == 0) {
            for (int g = 0; g < NGRP; ++g) {
                const int code = SCHED_[g];
                if (code == 7) {
                    for (int c = ci >> 1; c < CA; c += 2) {
                        while (lds_get(cnt + 32 + c) < 4u) __builtin_amdgcn_s_sleep(1);
                        __hip_atomic_fetch_add(ctl(dws, C_WCOL + rank * 16 + c), 2u, __ATOMIC_RELAXED, __HIP_MEMORY_SCOPE_SYSTEM);
                    }
                    STAMP(myws, 2);
                    continue;
                }
                while (lds_get(cnt + g) < (code < 4 ? 32u : code < 8 ? (unsigned)(CA * 2) : (unsigned)(32 - CA * 2))) __builtin_amdgcn_s_sleep(1);
                unsigned* ctr = code < 4 ? ctl(dws, C_XCNT + rank * JC + code) : ctl(dws, C_WCNT + rank * 8 + (code - 4));
                __hip_atomic_fetch_add(ctr, 1u, __ATOMIC_RELAXED, __HIP_MEMORY_SCOPE_SYSTEM);
            }
            STAMP(myws, 3);
            if (remote) {
                for (int tt = 0; tt < 8; ++tt) {
                    const int t = tt * NC + ci;
                    spin_ge<16>(ctl(myws, C_TILE + (2 * d + (t & 1)) * 16 + (t >> 1)), 8u * call, tmo, call);
                    __hip_atomic_store(cnt + 25, (unsigned)(tt + 1), __ATOMIC_RELAXED, __HIP_MEMORY_SCOPE_WORKGROUP);
                    if (tt == 0) STAMP(myws, 4);
                    if (tt == 6) STAMP(myws, 5);
                    if (tt == 7) STAMP(myws, 6);
                }
                while (lds_get(cnt + 24) < 128u) __builtin_amdgcn_s_sleep(1);
                __hip_atomic_fetch_add(ctl(dws, C_ODONE + rank), 8u, __ATOMIC_RELAXED, __HIP_MEMORY_SCOPE_SYSTEM);
                STAMP(myws, 7);
            } else if (ci == 0) {
                for (int s = 0; s < NR; ++s) if (s != rank) spin_ge<2>(ctl(myws, C_ODONE + s), 32u * call, tmo, call);
                __builtin_amdgcn_fence(__ATOMIC_ACQUIRE, "");
                if (__hip_atomic_load(tmo, __ATOMIC_RELAXED, __HIP_MEMORY_SCOPE_AGENT) == call) p.out[(size_t)rank * NS] = (bf16_t)0x7fc0u;
                STAMP(myws, 8);
            }
        }
        return;
    }

    LAS bf16_t* T = (LAS bf16_t*)(lds + wid * TW_BYTES);
    const int gw = ci * NDW + wid;
    if (wid == 0) STAMP(myws, 1);
    auto push_x = [&](int j, int g) {
        const g128* src = (const g128*)(p.x + j * KC) + (lane >> 4) * (KS / 8) + (lane & 15);
        const __amdgpu_buffer_rsrc_t dst = mk_rsrc(dws + OFF_A + par * STG + (size_t)(rank * JC + j) * CHB, (unsigned)CHB);
        stream_pieces<NW>(gw, 128,
            [&](int P, u32x4 (&v)[8]) {
#pragma unroll
                for (int u = 0; u < 8; ++u) v[u] = src[(size_t)(P * 32 + u * 4) * (KS / 8)]; },
            [&](int P, u32x4 (&v)[8]) {
#pragma unroll
                for (int u = 0; u < 8; ++u) st_sys(dst, (unsigned)(P * 512 + u * 64 + lane) * 16u, v[u]); },
            [&](int P) { if (lane == 0) lds_inc(cnt + g); });
    };
    auto push_w = [&](int part, int j, int g) {
        const g128* src = (const g128*)(p.w + (size_t)(j * KC) * NTOT + (size_t)d * NS) + (size_t)(lane >> 2) * (NTOT / 8) + (lane & 3);
        const __amdgpu_buffer_rsrc_t dst = mk_rsrc(dws + OFF_B + par * STG + (size_t)(rank * JC + j) * CHB, (unsigned)CHB);
        stream_pieces<NW>((part ? CA * 8 : 0) + gw, part ? 128 : CA * 8,
            [&](int P, u32x4 (&v)[8]) {
#pragma unroll
                for (int u = 0; u < 8; ++u) v[u] = src[(size_t)(u * 16) * (NTOT / 8) + P * 4]; },
            [&](int P, u32x4 (&v)[8]) {
#pragma unroll
                for (int u = 0; u < 8; ++u) *(LAS u32x4*)(T + (u * 16 + (lane >> 2)) * TP + (lane & 3) * 8) = v[u];
#pragma unroll
                for (int u = 0; u < 8; ++u) {
                    const int kb = (u & 3) * 4 + (lane >> 4), q = (lane & 15) >> 2, pp = lane & 3;
                    LAS bf16_t* a = T + (kb * 8 + q) * TP + (u >> 2) * 16 + pp * 4;
                    const s16x4 lo = __builtin_amdgcn_ds_read_tr16_b64_v4i16((LAS s16x4*)a);
                    const s16x4 hi = __builtin_amdgcn_ds_read_tr16_b64_v4i16((LAS s16x4*)(a + 4 * TP));
                    const bf16x8 o = __builtin_shufflevector(lo, hi, 0, 1, 2, 3, 4, 5, 6, 7);
                    st_sys(dst, (unsigned)(P * 512 + u * 64 + lane) * 16u, __builtin_bit_cast(u32x4, o));
                } },
            [&](int P) { if (lane == 0) lds_inc(cnt + (part == 0 && j == 3 ? 32 + (P >> 3) : g)); });
    };
    for (int g = 0; g < NGRP; ++g) {
        const int code = SCHED_[g];
        if (code < 4) push_x(code, g); else push_w((code - 4) >> 2, (code - 4) & 3, g);
    }
    if (remote) {
        const g128* src = (const g128*)(myws + OFF_O) + (size_t)(2 * d * 256 + (lane >> 5)) * (NS / 8) + (lane & 31);
        const __amdgpu_buffer_rsrc_t dst = mk_rsrc(p.out_all[d], MS * NTOT * 2);
        const unsigned doff = (unsigned)((lane >> 5) * (NTOT / 8) + rank * (NS / 8) + (lane & 31));
        stream_pieces<NDW>(wid, 128,
            [&](int P, u32x4 (&v)[8]) {
                const int t = (P >> 4) * NC + ci, row = (t & 1) * 256 + (P & 15) * 16, col = (t >> 1) * 32;
                while (lds_get(cnt + 25) <= (unsigned)(P >> 4)) __builtin_amdgcn_s_sleep(8);
                asm volatile("" ::: "memory");
#pragma unroll
                for (int u = 0; u < 8; ++u) v[u] = src[(size_t)(row + u * 2) * (NS / 8) + col]; },
            [&](int P, u32x4 (&v)[8]) {
                const int t = (P >> 4) * NC + ci, row = (t & 1) * 256 + (P & 15) * 16, col = (t >> 1) * 32;
#pragma unroll
                for (int u = 0; u < 8; ++u) st_sys(dst, (doff + (unsigned)((row + u * 2) * (NTOT / 8) + col)) * 16u, v[u]); },
            [&](int P) { if (lane == 0) lds_inc(cnt + 24); });
    }
}

__device__ void gemm_role(const Params& p, LAS unsigned char* lds, unsigned call, int xg, int idx) {
    const int tid = threadIdx.x, wid = __builtin_amdgcn_readfirstlane(tid >> 6), lane = tid & 63, wr = wid >> 2, wc = wid & 3, fr = lane & 15, fq = lane >> 4;
    const int rank = p.rank;
    char* myws = (char*)p.ws_all[rank];
    unsigned* tmo = ctl(myws, C_TMO);
    const unsigned par = call & 1u;
    const int pm = (xg & 3) * 4 + (idx & 3);
    const int pn0 = idx < 2 * CA ? (xg >> 2) * (CA / 2) + (CA / 2 - 1 - (idx >> 2)) : CA + (xg >> 2) * (7 - CA / 2) + ((idx - 2 * CA) >> 2), pn1 = 14 + (xg >> 2);
    const int nun = idx < 4 * RS ? 2 : 1;
    const int part = idx >> 2;
    const int q1 = part * (NCH / RS);

    unsigned voffA[2], voffB[2];
#pragma unroll
    for (int i = 0; i < 2; ++i) { int R, C; stage_rc(tid * 16 + i * 8192, R, C); const int Rb = (R & ~31) + perm32(R & 31);
        voffA[i] = (unsigned)(R * KC + C) * 2u; voffB[i] = (unsigned)(((Rb >> 4) * 16 + (C >> 3)) * 256 + (Rb & 15) * 16); }
    constexpr size_t kstep = BK * 2;
    constexpr size_t kstepB = 8 * 256;
    constexpr size_t hstep = (size_t)HALF * KC * 2;
    constexpr size_t tstep = 2 * hstep;
    const unsigned ldsw = (unsigned)wid * 1024u;
    const int aoff = lds_byte(wr * 64 + fr, fq * 8), boff = lds_byte(wc * 32 + fr, fq * 8);
#define SA(b, h) (((b) * 2 + (h)) * HTB)
#define SB(b, h) ((4 + (b) * 2 + (h)) * HTB)
#define STAGE(bufoff, gbase, voff) do { _Pragma("unroll") for (int _i = 0; _i < 2; ++_i) \
        __builtin_amdgcn_global_load_lds((const unsigned*)((const char*)(gbase) + (voff)[_i]), (LAS unsigned*)(lds + (bufoff) + ldsw + _i * 8192), 16, 0, 0); } while (0)
#define LDA(dst, b, h) do { _Pragma("unroll") for (int m = 0; m < 4; ++m) _Pragma("unroll") for (int k = 0; k < 2; ++k) dst[m][k] = *(const LAS bf16x8*)(lds + SA(b, h) + aoff + m * 2048 + k * 1024); } while (0)
#define LDB(dst, b, h) do { _Pragma("unroll") for (int n = 0; n < 2; ++n) _Pragma("unroll") for (int k = 0; k < 2; ++k) dst[n][k] = *(const LAS bf16x8*)(lds + SB(b, h) + boff + n * 2048 + k * 1024); } while (0)
#define MMA(ai, bj, At, Bt) do { __builtin_amdgcn_s_setprio(1); _Pragma("unroll") for (int m = 0; m < 4; ++m) _Pragma("unroll") for (int n = 0; n < 2; ++n) _Pragma("unroll") for (int k = 0; k < 2; ++k) \
        acc[ai][bj][m][n] = __builtin_amdgcn_mfma_f32_16x16x32_bf16(Bt[n][k], At[m][k], acc[ai][bj][m][n], 0, 0, 0); __builtin_amdgcn_s_setprio(0); } while (0)
#define WAIT_V(n) asm volatile("s_waitcnt vmcnt(" #n ")" ::: "memory")
#define WAIT_L(n) asm volatile("s_waitcnt lgkmcnt(" #n ")" ::: "memory")
#define BAR __builtin_amdgcn_s_barrier()
#define SCHED __builtin_amdgcn_sched_barrier(0)

    const char* Abase = myws + OFF_A + par * STG + (size_t)pm * tstep;
    const char* Bbase = myws + OFF_B + par * STG;
    auto chunk_of = [&](int q) { const int j = q >> 3, s = q & 7; return ((rank + s) & 7) * JC + j; };

    int known = 0;
    auto ready = [&](int h, int q) {
        if (wid == 0 && known <= q) {
            const unsigned long long t0 = __builtin_amdgcn_s_memrealtime();
            const int kc = chunk_of(lane & 31);
            unsigned* fx = ctl(myws, C_XCNT + kc);
            const int pnh = h ? pn1 : pn0;
            unsigned* fw = pnh < CA && (kc & 3) == 3 ? ctl(myws, C_WCOL + (kc >> 2) * 16 + pnh)
                                                     : ctl(myws, C_WCNT + (kc >> 2) * 8 + (pnh >= CA ? 4 : 0) + (kc & 3));
            const unsigned need = (unsigned)NC * call;
            for (;;) {
                const unsigned vx = __hip_atomic_load(fx, __ATOMIC_RELAXED, __HIP_MEMORY_SCOPE_SYSTEM);
                const unsigned vw = __hip_atomic_load(fw, __ATOMIC_RELAXED, __HIP_MEMORY_SCOPE_SYSTEM);
                const unsigned long long ok = __ballot(vx >= need && vw >= need) | 0xffffffff00000000ull;
                known = (~ok == 0ull) ? 32 : __builtin_ctzll(~ok);
                if (known > q) break;
                if (__builtin_amdgcn_readfirstlane(__hip_atomic_load(tmo, __ATOMIC_RELAXED, __HIP_MEMORY_SCOPE_AGENT)) == call
                    || __builtin_amdgcn_s_memrealtime() - t0 > DEADLINE_TICKS) {
                    if (lane == 0) __hip_atomic_store(tmo, call, __ATOMIC_RELAXED, __HIP_MEMORY_SCOPE_AGENT);
                    known = 32; break;
                }
                __builtin_amdgcn_s_sleep(4);
            }
        }
        asm volatile("" ::: "memory"); BAR; asm volatile("" ::: "memory");
    };

    f32x4 acc[2][2][4][2];
#pragma unroll
    for (int a = 0; a < 2; ++a)
#pragma unroll
        for (int b = 0; b < 2; ++b)
#pragma unroll
            for (int m = 0; m < 4; ++m)
#pragma unroll
                for (int n = 0; n < 2; ++n) acc[a][b][m][n] = (f32x4){0.f, 0.f, 0.f, 0.f};
    bf16x8 At[4][2], B0[2][2], B1[2][2];

    {
        const char* cA = Abase + (size_t)chunk_of(0) * CHB;
        const char* cB = Bbase + (size_t)chunk_of(0) * CHB + (size_t)pn0 * tstep;
        STAMP(myws, 1);
        ready(0, 0);
        STAMP(myws, 2);
        STAGE(SB(0, 0), cB, voffB); STAGE(SA(0, 0), cA, voffA); STAGE(SB(0, 1), cB + hstep, voffB); STAGE(SA(0, 1), cA + hstep, voffA);
        if (wr == 1) BAR;
        WAIT_V(4); BAR;
        STAGE(SB(1, 0), cB + kstepB, voffB); STAGE(SA(1, 0), cA + kstep, voffA); STAGE(SB(1, 1), cB + hstep + kstepB, voffB);
        WAIT_V(6); BAR;
    }
    for (int h = 0; h < nun; ++h) {
        const int pn = h ? pn1 : pn0;
        const int qb = h ? q1 : 0, qe = h ? q1 + NCH / RS : NCH;
        for (int q = qb; q < qe; ++q) {
            const bool last = (q == qe - 1);
            const bool nxt = last && h + 1 < nun;
            const int hn = nxt ? h + 1 : h, qn = nxt ? q1 : last ? qb : q + 1;
            const char* a1 = Abase + (size_t)chunk_of(q) * CHB + kstep;
            const char* a2 = Abase + (size_t)chunk_of(qn) * CHB;
            const char* b2 = Bbase + (size_t)chunk_of(qn) * CHB + (size_t)(hn ? pn1 : pn0) * tstep;
            const char* a3 = a2 + kstep; const char* b3 = b2 + kstepB;
            if (nxt) known = 0;
            if (wid == 0) STAMP(myws, 16 + (h * 32 + q) * 2);
            ready(hn, qn);
            if (wid == 0) STAMP(myws, 17 + (h * 32 + q) * 2);
            LDB(B0, 0, 0); SCHED; LDA(At, 0, 0); STAGE(SA(1, 1), a1 + hstep, voffA);
            WAIT_L(8); BAR; WAIT_L(0); MMA(0, 0, At, B0); BAR; SCHED;
            LDB(B1, 0, 1); STAGE(SB(0, 0), b2, voffB);
            BAR; WAIT_L(0); MMA(0, 1, At, B1); BAR;
            LDA(At, 0, 1); STAGE(SA(0, 0), a2, voffA);
            BAR; WAIT_L(0); MMA(1, 0, At, B0); BAR; SCHED;
            STAGE(SB(0, 1), b2 + hstep, voffB);
            WAIT_V(6); BAR; MMA(1, 1, At, B1); BAR;
            LDB(B0, 1, 0); SCHED; LDA(At, 1, 0); STAGE(SA(0, 1), a2 + hstep, voffA);
            WAIT_L(8); BAR; WAIT_L(0); MMA(0, 0, At, B0); BAR; SCHED;
            LDB(B1, 1, 1); STAGE(SB(1, 0), b3, voffB);
            BAR; WAIT_L(0); MMA(0, 1, At, B1); BAR;
            LDA(At, 1, 1); STAGE(SA(1, 0), a3, voffA);
            BAR; WAIT_L(0); MMA(1, 0, At, B0); BAR; SCHED;
            STAGE(SB(1, 1), b3 + hstep, voffB);
            WAIT_V(6); BAR; MMA(1, 1, At, B1); BAR;
        }
        STAMP(myws, 3 + 2 * h);
        if (h == 1) {
            const int rt = (pm * 2 + (pn1 - 14)) * (RS - 1);
            if (part) {
                const __amdgpu_buffer_rsrc_t rs = mk_rsrc(myws + OFF_P + (size_t)(rt + part - 1) * 262144, 262144);
#pragma unroll
                for (int i = 0; i < 32; ++i)
                    __builtin_amdgcn_raw_buffer_store_b128(__builtin_bit_cast(u32x4, acc[i >> 4][(i >> 3) & 1][(i >> 1) & 3][i & 1]), rs, tid * 16, i * 8192, 16);
                asm volatile("s_waitcnt vmcnt(0)" ::: "memory");
                if (lane == 0) __hip_atomic_fetch_add(ctl(myws, C_PART + rt + part - 1), 1u, __ATOMIC_RELAXED, __HIP_MEMORY_SCOPE_AGENT);
                break;
            }
            for (int pt = 0; pt < RS - 1; ++pt) {
                if (lane == 0) spin_ge<4>(ctl(myws, C_PART + rt + pt), 8u * call, tmo, call);
                asm volatile("" ::: "memory");
                const __amdgpu_buffer_rsrc_t rs = mk_rsrc(myws + OFF_P + (size_t)(rt + pt) * 262144, 262144);
#pragma unroll
                for (int i = 0; i < 32; ++i) { acc[i >> 4][(i >> 3) & 1][(i >> 1) & 3][i & 1] += __builtin_bit_cast(f32x4, __builtin_amdgcn_raw_buffer_load_b128(rs, tid * 16, i * 8192, 0));
                    if ((i & 7) == 7) asm volatile("" ::: "memory"); }
            }
        }
        {
            const bool own = (pm >> 1) == rank;
            const int row0 = wr * 64 + fr, col0 = wc * 32 + 8 * fq;
            if (own) {
                bf16_t* base = p.out + (size_t)((pm & 1) * 256) * NTOT + (size_t)rank * NS + pn * 256;
#pragma unroll
                for (int ai = 0; ai < 2; ++ai)
#pragma unroll
                    for (int m = 0; m < 4; ++m) { bf16_t* rowp = base + (size_t)(row0 + ai * HALF + m * 16) * NTOT + col0;
#pragma unroll
                        for (int bj = 0; bj < 2; ++bj) *(u32x4*)(rowp + bj * HALF) = pack8(acc[ai][bj][m][0], acc[ai][bj][m][1]); }
            } else {
                const __amdgpu_buffer_rsrc_t rs = mk_rsrc(myws + OFF_O + ((size_t)(pm * 256) * NS + pn * 256) * 2, 256 * NS * 2);
#pragma unroll
                for (int ai = 0; ai < 2; ++ai)
#pragma unroll
                    for (int m = 0; m < 4; ++m) { const int ro = ((row0 + ai * HALF + m * 16) * NS + col0) * 2;
#pragma unroll
                        for (int bj = 0; bj < 2; ++bj) __builtin_amdgcn_raw_buffer_store_b128(pack8(acc[ai][bj][m][0], acc[ai][bj][m][1]), rs, ro + bj * HALF * 2, 0, 16); }
                asm volatile("s_waitcnt vmcnt(0)" ::: "memory");
                if (lane == 0) __hip_atomic_fetch_add(ctl(myws, C_TILE + pm * 16 + pn), 1u, __ATOMIC_RELAXED, __HIP_MEMORY_SCOPE_AGENT);
            }
        }
        STAMP(myws, 4 + 2 * h);
#pragma unroll
        for (int a = 0; a < 2; ++a)
#pragma unroll
            for (int b = 0; b < 2; ++b)
#pragma unroll
                for (int m = 0; m < 4; ++m)
#pragma unroll
                    for (int n = 0; n < 2; ++n) acc[a][b][m][n] = (f32x4){0.f, 0.f, 0.f, 0.f};
    }
    WAIT_V(0);
    if (wr == 0) BAR;
    BAR;
#undef SA
#undef SB
#undef STAGE
#undef LDA
#undef LDB
#undef MMA
#undef WAIT_V
#undef WAIT_L
#undef BAR
#undef SCHED
}

__global__ __launch_bounds__(512, 2) void fused_kernel(Params p) {
    extern __shared__ __attribute__((aligned(16))) unsigned char shm[];
    LAS unsigned char* lds = (LAS unsigned char*)shm;
    LAS unsigned* lw = (LAS unsigned*)(lds + STAGE_BYTES);
    const int grp = (int)blockIdx.x & 7, slot = (int)blockIdx.x >> 3;
    STAMP(p.ws_all[p.rank], 0);
    if (threadIdx.x == 0) {
        const unsigned t = __hip_atomic_fetch_add(ctl(p.ws_all[p.rank], C_TICKET), 1u, __ATOMIC_RELAXED, __HIP_MEMORY_SCOPE_AGENT);
        lw[0] = t / (unsigned)NBLK + 1u;
        __builtin_amdgcn_fence(__ATOMIC_ACQUIRE, "");
        asm volatile("s_waitcnt vmcnt(0)" ::: "memory");
    }
    __syncthreads();
    const unsigned call = __builtin_amdgcn_readfirstlane(lw[0]);
    __syncthreads();
    if (slot < NC) comm_role(p, lds, call, (p.rank + grp) & 7, slot);
    else gemm_role(p, lds, call, grp, slot - NC);
}

}

extern "C" void kernel_launch(int rank, int nranks, ncclComm_t comm,
                              hipStream_t stream,
                              void* const* d_in, const int* in_sizes, int n_in,
                              void* d_out, int out_size,
                              const int* in_dtypes, int out_dtype,
                              void* const* d_out_all,
                              void* const* d_ws_all, size_t ws_bytes) {
    if (nranks != NR || ws_bytes < WS_NEED || (unsigned)rank >= (unsigned)NR) { fprintf(stderr, "unexpected configuration\n"); return; }
    static bool attr[64] = {};
    int dev = 0; HIP_CHECK(hipGetDevice(&dev));
    if (!attr[dev & 63]) { HIP_CHECK(hipFuncSetAttribute((const void*)fused_kernel, hipFuncAttributeMaxDynamicSharedMemorySize, LDS_BYTES)); attr[dev & 63] = true; }
    Params p;
    p.x = (const bf16_t*)d_in[0]; p.w = (const bf16_t*)d_in[1]; p.out = (bf16_t*)d_out;
    p.out_all = d_out_all; p.ws_all = d_ws_all; p.rank = rank;
    hipLaunchKernelGGL(fused_kernel, dim3(NBLK), dim3(512), LDS_BYTES, stream, p);
    HIP_CHECK(hipGetLastError());
}
```

```cpp
#include <hip/hip_runtime.h>
#include <rccl/rccl.h>
#include <cstdio>

#define HIP_CHECK(expr) do { hipError_t _e = (expr); if (_e != hipSuccess) { \
    fprintf(stderr, "HIP error %s at %s:%d\n", hipGetErrorString(_e), __FILE__, __LINE__); \
    return; } } while (0)

namespace {

#define LAS __attribute__((address_space(3)))
#define GAS __attribute__((address_space(1)))
typedef unsigned short bf16_t;
typedef short bf16x8 __attribute__((ext_vector_type(8)));
typedef short s16x4 __attribute__((ext_vector_type(4)));
typedef float f32x4 __attribute__((ext_vector_type(4)));
typedef unsigned u32x4 __attribute__((ext_vector_type(4)));
typedef GAS u32x4 g128;

constexpr int NR = 8;
constexpr int MTOT = 4096, KTOT = 4096, NTOT = 32768;
constexpr int KS = KTOT / NR;
constexpr int NS = NTOT / NR;
constexpr int MS = MTOT / NR;
constexpr int KC = 128;
constexpr int JC = KS / KC;
constexpr int NCH = KTOT / KC;
constexpr int NC = 4;
constexpr int NG = 28;
constexpr int NBLK = 256;
constexpr int CA = 10;

constexpr size_t CTL_BYTES = 256 * 1024;
constexpr size_t CHB = (size_t)4096 * KC * 2;
constexpr size_t STG = (size_t)NCH * CHB;
constexpr size_t OFF_A = CTL_BYTES;
constexpr size_t OFF_B = OFF_A + 2 * STG;
constexpr size_t OFF_O = OFF_B + 2 * STG;
constexpr int RS = 2;
constexpr size_t OFF_P = OFF_O + (size_t)MTOT * NS * 2;
constexpr size_t WS_NEED = OFF_P + (size_t)32 * (RS - 1) * 256 * 256 * 4;
constexpr int CW = 64;
constexpr int C_TICKET = 0;
constexpr int C_TMO = 1;
constexpr int C_XCNT = 2;
constexpr int C_WCNT = C_XCNT + NR * JC;
constexpr int C_WCOL = C_WCNT + NR * 8;
constexpr int C_ODONE = C_WCOL + NR * 16;
constexpr int C_TILE = C_ODONE + NR;
constexpr int C_PART = C_TILE + 256;
constexpr int C_END = C_PART + 32 * 3;
static_assert((size_t)C_END * CW * 4 <= CTL_BYTES, "ctl");

constexpr unsigned long long DEADLINE_TICKS = 50000000ull;

constexpr int BK = 64, HALF = 128, HTB = HALF * BK * 2, STAGE_BYTES = 8 * HTB;
constexpr int LDS_BYTES = STAGE_BYTES + 1024;

__device__ __forceinline__ int lds_byte(int r, int c) { const int st = (r >> 4) * 2 + (c >> 5), rr = r & 15, cc = c & 31, ob = rr * 64 + cc * 2; return st * 1024 + (ob ^ (((ob >> 9) & 1) << 5)); }
__device__ __forceinline__ void stage_rc(int b, int& R, int& C) { const int st = b / 1024, sb = b % 1024, swz = sb ^ (((sb >> 9) & 1) << 5); R = (st >> 1) * 16 + swz / 64; C = (st & 1) * 32 + (swz % 64) / 2; }
__device__ __forceinline__ int perm32(int rho) { const int n = rho >> 4, i = rho & 15; return 8 * (i >> 2) + 4 * n + (i & 3); }
__device__ __forceinline__ unsigned cvt_pk_bf16(float lo, float hi) { unsigned r; asm volatile("v_cvt_pk_bf16_f32 %0, %1, %2" : "=v"(r) : "v"(lo), "v"(hi)); return r; }
__device__ __forceinline__ u32x4 pack8(f32x4 a, f32x4 b) {
    const f32x4 z = {0.f, 0.f, 0.f, 0.f};
    a = __builtin_elementwise_max(a, z); b = __builtin_elementwise_max(b, z);
    u32x4 w; w.x = cvt_pk_bf16(a[0], a[1]); w.y = cvt_pk_bf16(a[2], a[3]); w.z = cvt_pk_bf16(b[0], b[1]); w.w = cvt_pk_bf16(b[2], b[3]);
    return w;
}

struct Params {
    const bf16_t* x;
    const bf16_t* w;
    bf16_t* out;
    void* const* out_all;
    void* const* ws_all;
    int rank;
};

__device__ __forceinline__ unsigned* ctl(void* ws, int line) { return (unsigned*)ws + (size_t)line * CW; }
#define STAMP(ws, k) do {} while (0)

template <int SLEEP>
__device__ __forceinline__ void spin_ge(unsigned* flag, unsigned need, unsigned* tmo, unsigned call) {
    const unsigned long long t0 = __builtin_amdgcn_s_memrealtime();
    while (__hip_atomic_load(flag, __ATOMIC_RELAXED, __HIP_MEMORY_SCOPE_SYSTEM) < need) {
        if (__hip_atomic_load(tmo, __ATOMIC_RELAXED, __HIP_MEMORY_SCOPE_AGENT) == call
            || __builtin_amdgcn_s_memrealtime() - t0 > DEADLINE_TICKS) {
            __hip_atomic_store(tmo, call, __ATOMIC_RELAXED, __HIP_MEMORY_SCOPE_AGENT);
            break;
        }
        __builtin_amdgcn_s_sleep(SLEEP);
    }
}

__device__ __forceinline__ __amdgpu_buffer_rsrc_t mk_rsrc(const void* base, unsigned bytes) {
    const unsigned long long a = (unsigned long long)base;
    const unsigned lo = __builtin_amdgcn_readfirstlane((unsigned)a), hi = __builtin_amdgcn_readfirstlane((unsigned)(a >> 32));
    return __builtin_amdgcn_make_buffer_rsrc((void*)(((unsigned long long)hi << 32) | lo), (short)0, (int)bytes, 0x00020000);
}
__device__ __forceinline__ void st_sys(__amdgpu_buffer_rsrc_t rs, unsigned off, u32x4 v) { __builtin_amdgcn_raw_buffer_store_b128(v, rs, (int)off, 0, 17); }

constexpr int NDW = 4;
constexpr int NW = NC * NDW;
constexpr int NGRP = 12;
__device__ constexpr int SCHED_[NGRP] = {0, 4, 1, 5, 2, 6, 8, 3, 7, 9, 10, 11};
constexpr int TP = 32;
constexpr int TW_BYTES = 128 * TP * 2;

__device__ __forceinline__ void lds_inc(LAS unsigned* w) { __hip_atomic_fetch_add(w, 1u, __ATOMIC_RELAXED, __HIP_MEMORY_SCOPE_WORKGROUP); }
__device__ __forceinline__ unsigned lds_get(LAS unsigned* w) { return __hip_atomic_load(w, __ATOMIC_RELAXED, __HIP_MEMORY_SCOPE_WORKGROUP); }

template <int S, class LD, class ST, class SG>
__device__ __forceinline__ void stream_pieces(int first, int np, LD ld, ST st, SG sig) {
    u32x4 a[8], b[8];
    int k = first;
    ld(k, a);
    if (k + S >= np) { st(k, a); asm volatile("s_waitcnt vmcnt(0)" ::: "memory"); sig(k); return; }
    ld(k + S, b); st(k, a);
    for (; k + 3 * S < np; k += 2 * S) {
        ld(k + 2 * S, a); st(k + S, b);
        asm volatile("s_waitcnt vmcnt(16)" ::: "memory"); sig(k);
        ld(k + 3 * S, b); st(k + 2 * S, a);
        asm volatile("s_waitcnt vmcnt(16)" ::: "memory"); sig(k + S);
    }
    if (k + 2 * S < np) {
        ld(k + 2 * S, a); st(k + S, b);
        asm volatile("s_waitcnt vmcnt(16)" ::: "memory"); sig(k);
        st(k + 2 * S, a);
        asm volatile("s_waitcnt vmcnt(8)" ::: "memory"); sig(k + S);
        asm volatile("s_waitcnt vmcnt(0)" ::: "memory"); sig(k + 2 * S);
    } else {
        st(k + S, b);
        asm volatile("s_waitcnt vmcnt(8)" ::: "memory"); sig(k);
        asm volatile("s_waitcnt vmcnt(0)" ::: "memory"); sig(k + S);
    }
}

__device__ void comm_role(const Params& p, LAS unsigned char* lds, unsigned call, int d, int ci) {
    const int tid = threadIdx.x, wid = __builtin_amdgcn_readfirstlane(tid >> 6), lane = tid & 63;
    const int rank = p.rank;
    char* myws = (char*)p.ws_all[rank];
    char* dws = (char*)p.ws_all[d];
    const unsigned par = call & 1u;
    unsigned* tmo = ctl(myws, C_TMO);
    LAS unsigned* cnt = (LAS unsigned*)(lds + NDW * TW_BYTES);
    if (tid < 64) cnt[tid] = 0u;
    __syncthreads();
    const bool remote = d != rank;
    if (wid > NDW) return;

    if (wid == NDW) {
        if (lane == 0) {
            for (int g = 0; g < NGRP; ++g) {
                const int code = SCHED_[g];
                if (code == 7) {
                    for (int c = ci >> 1; c < CA; c += 2) {
                        while (lds_get(cnt + 32 + c) < 4u) __builtin_amdgcn_s_sleep(1);
                        __hip_atomic_fetch_add(ctl(dws, C_WCOL + rank * 16 + c), 2u, __ATOMIC_RELAXED, __HIP_MEMORY_SCOPE_SYSTEM);
                    }
                    STAMP(myws, 2);
                    continue;
                }
                while (lds_get(cnt + g) < (code < 4 ? 32u : code < 8 ? (unsigned)(CA * 2) : (unsigned)(32 - CA * 2))) __builtin_amdgcn_s_sleep(1);
                unsigned* ctr = code < 4 ? ctl(dws, C_XCNT + rank * JC + code) : ctl(dws, C_WCNT + rank * 8 + (code - 4));
                __hip_atomic_fetch_add(ctr, 1u, __ATOMIC_RELAXED, __HIP_MEMORY_SCOPE_SYSTEM);
            }
            STAMP(myws, 3);
            if (remote) {
                for (int tt = 0; tt < 8; ++tt) {
                    const int t = tt * NC + ci;
                    spin_ge<16>(ctl(myws, C_TILE + (2 * d + (t & 1)) * 16 + (t >> 1)), 8u * call, tmo, call);
                    __hip_atomic_store(cnt + 25, (unsigned)(tt + 1), __ATOMIC_RELAXED, __HIP_MEMORY_SCOPE_WORKGROUP);
                    if (tt == 0) STAMP(myws, 4);
                    if (tt == 6) STAMP(myws, 5);
                    if (tt == 7) STAMP(myws, 6);
                }
                while (lds_get(cnt + 24) < 128u) __builtin_amdgcn_s_sleep(1);
                __hip_atomic_fetch_add(ctl(dws, C_ODONE + rank), 8u, __ATOMIC_RELAXED, __HIP_MEMORY_SCOPE_SYSTEM);
                STAMP(myws, 7);
            } else if (ci == 0) {
                for (int s = 0; s < NR; ++s) if (s != rank) spin_ge<2>(ctl(myws, C_ODONE + s), 32u * call, tmo, call);
                __builtin_amdgcn_fence(__ATOMIC_ACQUIRE, "");
                if (__hip_atomic_load(tmo, __ATOMIC_RELAXED, __HIP_MEMORY_SCOPE_AGENT) == call) p.out[(size_t)rank * NS] = (bf16_t)0x7fc0u;
                STAMP(myws, 8);
            }
        }
        return;
    }

    LAS bf16_t* T = (LAS bf16_t*)(lds + wid * TW_BYTES);
    const int gw = ci * NDW + wid;
    if (wid == 0) STAMP(myws, 1);
    auto push_x = [&](int j, int g) {
        const g128* src = (const g128*)(p.x + j * KC) + (lane >> 4) * (KS / 8) + (lane & 15);
        const __amdgpu_buffer_rsrc_t dst = mk_rsrc(dws + OFF_A + par * STG + (size_t)(rank * JC + j) * CHB, (unsigned)CHB);
        stream_pieces<NW>(gw, 128,
            [&](int P, u32x4 (&v)[8]) {
#pragma unroll
                for (int u = 0; u < 8; ++u) v[u] = src[(size_t)(P * 32 + u * 4) * (KS / 8)]; },
            [&](int P, u32x4 (&v)[8]) {
#pragma unroll
                for (int u = 0; u < 8; ++u) st_sys(dst, (unsigned)(P * 512 + u * 64 + lane) * 16u, v[u]); },
            [&](int P) { if (lane == 0) lds_inc(cnt + g); });
    };
    auto push_w = [&](int part, int j, int g) {
        const g128* src = (const g128*)(p.w + (size_t)(j * KC) * NTOT + (size_t)d * NS) + (size_t)(lane >> 2) * (NTOT / 8) + (lane & 3);
        const __amdgpu_buffer_rsrc_t dst = mk_rsrc(dws + OFF_B + par * STG + (size_t)(rank * JC + j) * CHB, (unsigned)CHB);
        stream_pieces<NW>((part ? CA * 8 : 0) + gw, part ? 128 : CA * 8,
            [&](int P, u32x4 (&v)[8]) {
#pragma unroll
                for (int u = 0; u < 8; ++u) v[u] = src[(size_t)(u * 16) * (NTOT / 8) + P * 4]; },
            [&](int P, u32x4 (&v)[8]) {
#pragma unroll
                for (int u = 0; u < 8; ++u) *(LAS u32x4*)(T + (u * 16 + (lane >> 2)) * TP + (lane & 3) * 8) = v[u];
#pragma unroll
                for (int u = 0; u < 8; ++u) {
                    const int kb = (u & 3) * 4 + (lane >> 4), q = (lane & 15) >> 2, pp = lane & 3;
                    LAS bf16_t* a = T + (kb * 8 + q) * TP + (u >> 2) * 16 + pp * 4;
                    const s16x4 lo = __builtin_amdgcn_ds_read_tr16_b64_v4i16((LAS s16x4*)a);
                    const s16x4 hi = __builtin_amdgcn_ds_read_tr16_b64_v4i16((LAS s16x4*)(a + 4 * TP));
                    const bf16x8 o = __builtin_shufflevector(lo, hi, 0, 1, 2, 3, 4, 5, 6, 7);
                    st_sys(dst, (unsigned)(P * 512 + u * 64 + lane) * 16u, __builtin_bit_cast(u32x4, o));
                } },
            [&](int P) { if (lane == 0) lds_inc(cnt + (part == 0 && j == 3 ? 32 + (P >> 3) : g)); });
    };
    for (int g = 0; g < NGRP; ++g) {
        const int code = SCHED_[g];
        if (code < 4) push_x(code, g); else push_w((code - 4) >> 2, (code - 4) & 3, g);
    }
    if (remote) {
        const g128* src = (const g128*)(myws + OFF_O) + (size_t)(2 * d * 256 + (lane >> 5)) * (NS / 8) + (lane & 31);
        const __amdgpu_buffer_rsrc_t dst = mk_rsrc(p.out_all[d], MS * NTOT * 2);
        const unsigned doff = (unsigned)((lane >> 5) * (NTOT / 8) + rank * (NS / 8) + (lane & 31));
        stream_pieces<NDW>(wid, 128,
            [&](int P, u32x4 (&v)[8]) {
                const int t = (P >> 4) * NC + ci, row = (t & 1) * 256 + (P & 15) * 16, col = (t >> 1) * 32;
                while (lds_get(cnt + 25) <= (unsigned)(P >> 4)) __builtin_amdgcn_s_sleep(8);
                asm volatile("" ::: "memory");
#pragma unroll
                for (int u = 0; u < 8; ++u) v[u] = src[(size_t)(row + u * 2) * (NS / 8) + col]; },
            [&](int P, u32x4 (&v)[8]) {
                const int t = (P >> 4) * NC + ci, row = (t & 1) * 256 + (P & 15) * 16, col = (t >> 1) * 32;
#pragma unroll
                for (int u = 0; u < 8; ++u) st_sys(dst, (doff + (unsigned)((row + u * 2) * (NTOT / 8) + col)) * 16u, v[u]); },
            [&](int P) { if (lane == 0) lds_inc(cnt + 24); });
    }
}

__device__ void gemm_role(const Params& p, LAS unsigned char* lds, unsigned call, int xg, int idx) {
    const int tid = threadIdx.x, wid = __builtin_amdgcn_readfirstlane(tid >> 6), lane = tid & 63, wr = wid >> 2, wc = wid & 3, fr = lane & 15, fq = lane >> 4;
    const int rank = p.rank;
    char* myws = (char*)p.ws_all[rank];
    unsigned* tmo = ctl(myws, C_TMO);
    const unsigned par = call & 1u;
    const int pm = (xg & 3) * 4 + (idx & 3);
    const int pn0 = idx < 2 * CA ? (xg >> 2) * (CA / 2) + (CA / 2 - 1 - (idx >> 2)) : CA + (xg >> 2) * (7 - CA / 2) + ((idx - 2 * CA) >> 2), pn1 = 14 + (xg >> 2);
    const int nun = idx < 4 * RS ? 2 : 1;
    const int part = idx >> 2;
    const int q1 = part * (NCH / RS);

    unsigned voffA[2], voffB[2];
#pragma unroll
    for (int i = 0; i < 2; ++i) { int R, C; stage_rc(tid * 16 + i * 8192, R, C); const int Rb = (R & ~31) + perm32(R & 31);
        voffA[i] = (unsigned)(R * KC + C) * 2u; voffB[i] = (unsigned)(((Rb >> 4) * 16 + (C >> 3)) * 256 + (Rb & 15) * 16); }
    constexpr size_t kstep = BK * 2;
    constexpr size_t kstepB = 8 * 256;
    constexpr size_t hstep = (size_t)HALF * KC * 2;
    constexpr size_t tstep = 2 * hstep;
    const unsigned ldsw = (unsigned)wid * 1024u;
    const int aoff = lds_byte(wr * 64 + fr, fq * 8), boff = lds_byte(wc * 32 + fr, fq * 8);
#define SA(b, h) (((b) * 2 + (h)) * HTB)
#define SB(b, h) ((4 + (b) * 2 + (h)) * HTB)
#define STAGE(bufoff, gbase, voff) do { _Pragma("unroll") for (int _i = 0; _i < 2; ++_i) \
        __builtin_amdgcn_global_load_lds((const unsigned*)((const char*)(gbase) + (voff)[_i]), (LAS unsigned*)(lds + (bufoff) + ldsw + _i * 8192), 16, 0, 0); } while (0)
#define LDA(dst, b, h) do { _Pragma("unroll") for (int m = 0; m < 4; ++m) _Pragma("unroll") for (int k = 0; k < 2; ++k) dst[m][k] = *(const LAS bf16x8*)(lds + SA(b, h) + aoff + m * 2048 + k * 1024); } while (0)
#define LDB(dst, b, h) do { _Pragma("unroll") for (int n = 0; n < 2; ++n) _Pragma("unroll") for (int k = 0; k < 2; ++k) dst[n][k] = *(const LAS bf16x8*)(lds + SB(b, h) + boff + n * 2048 + k * 1024); } while (0)
#define MMA(ai, bj, At, Bt) do { __builtin_amdgcn_s_setprio(1); _Pragma("unroll") for (int m = 0; m < 4; ++m) _Pragma("unroll") for (int n = 0; n < 2; ++n) _Pragma("unroll") for (int k = 0; k < 2; ++k) \
        acc[ai][bj][m][n] = __builtin_amdgcn_mfma_f32_16x16x32_bf16(Bt[n][k], At[m][k], acc[ai][bj][m][n], 0, 0, 0); __builtin_amdgcn_s_setprio(0); } while (0)
#define WAIT_V(n) asm volatile("s_waitcnt vmcnt(" #n ")" ::: "memory")
#define WAIT_L(n) asm volatile("s_waitcnt lgkmcnt(" #n ")" ::: "memory")
#define BAR __builtin_amdgcn_s_barrier()
#define SCHED __builtin_amdgcn_sched_barrier(0)

    const char* Abase = myws + OFF_A + par * STG + (size_t)pm * tstep;
    const char* Bbase = myws + OFF_B + par * STG;
    auto chunk_of = [&](int q) { const int j = q >> 3, s = q & 7; return ((rank + s) & 7) * JC + j; };

    int known = 0;
    auto ready = [&](int h, int q) {
        if (wid == 0 && known <= q) {
            const unsigned long long t0 = __builtin_amdgcn_s_memrealtime();
            const int kc = chunk_of(lane & 31);
            unsigned* fx = ctl(myws, C_XCNT + kc);
            const int pnh = h ? pn1 : pn0;
            unsigned* fw = pnh < CA && (kc & 3) == 3 ? ctl(myws, C_WCOL + (kc >> 2) * 16 + pnh)
                                                     : ctl(myws, C_WCNT + (kc >> 2) * 8 + (pnh >= CA ? 4 : 0) + (kc & 3));
            const unsigned need = (unsigned)NC * call;
            for (;;) {
                const unsigned vx = __hip_atomic_load(fx, __ATOMIC_RELAXED, __HIP_MEMORY_SCOPE_SYSTEM);
                const unsigned vw = __hip_atomic_load(fw, __ATOMIC_RELAXED, __HIP_MEMORY_SCOPE_SYSTEM);
                const unsigned long long ok = __ballot(vx >= need && vw >= need) | 0xffffffff00000000ull;
                known = (~ok == 0ull) ? 32 : __builtin_ctzll(~ok);
                if (known > q) break;
                if (__builtin_amdgcn_readfirstlane(__hip_atomic_load(tmo, __ATOMIC_RELAXED, __HIP_MEMORY_SCOPE_AGENT)) == call
                    || __builtin_amdgcn_s_memrealtime() - t0 > DEADLINE_TICKS) {
                    if (lane == 0) __hip_atomic_store(tmo, call, __ATOMIC_RELAXED, __HIP_MEMORY_SCOPE_AGENT);
                    known = 32; break;
                }
                __builtin_amdgcn_s_sleep(4);
            }
        }
        asm volatile("" ::: "memory"); BAR; asm volatile("" ::: "memory");
    };

    f32x4 acc[2][2][4][2];
#pragma unroll
    for (int a = 0; a < 2; ++a)
#pragma unroll
        for (int b = 0; b < 2; ++b)
#pragma unroll
            for (int m = 0; m < 4; ++m)
#pragma unroll
                for (int n = 0; n < 2; ++n) acc[a][b][m][n] = (f32x4){0.f, 0.f, 0.f, 0.f};
    bf16x8 At[4][2], B0[2][2], B1[2][2];

    {
        const char* cA = Abase + (size_t)chunk_of(0) * CHB;
        const char* cB = Bbase + (size_t)chunk_of(0) * CHB + (size_t)pn0 * tstep;
        STAMP(myws, 1);
        ready(0, 0);
        STAMP(myws, 2);
        STAGE(SB(0, 0), cB, voffB); STAGE(SA(0, 0), cA, voffA); STAGE(SB(0, 1), cB + hstep, voffB); STAGE(SA(0, 1), cA + hstep, voffA);
        if (wr == 1) BAR;
        WAIT_V(4); BAR;
        STAGE(SB(1, 0), cB + kstepB, voffB); STAGE(SA(1, 0), cA + kstep, voffA); STAGE(SB(1, 1), cB + hstep + kstepB, voffB);
        WAIT_V(6); BAR;
    }
    for (int h = 0; h < nun; ++h) {
        const int pn = h ? pn1 : pn0;
        const int qb = h ? q1 : 0, qe = h ? q1 + NCH / RS : NCH;
        for (int q = qb; q < qe; ++q) {
            const bool last = (q == qe - 1);
            const bool nxt = last && h + 1 < nun;
            const int hn = nxt ? h + 1 : h, qn = nxt ? q1 : last ? qb : q + 1;
            const char* a1 = Abase + (size_t)chunk_of(q) * CHB + kstep;
            const char* a2 = Abase + (size_t)chunk_of(qn) * CHB;
            const char* b2 = Bbase + (size_t)chunk_of(qn) * CHB + (size_t)(hn ? pn1 : pn0) * tstep;
            const char* a3 = a2 + kstep; const char* b3 = b2 + kstepB;
            if (nxt) known = 0;
            if (wid == 0) STAMP(myws, 16 + (h * 32 + q) * 2);
            ready(hn, qn);
            if (wid == 0) STAMP(myws, 17 + (h * 32 + q) * 2);
            LDB(B0, 0, 0); SCHED; LDA(At, 0, 0); STAGE(SA(1, 1), a1 + hstep, voffA);
            WAIT_L(8); BAR; WAIT_L(0); MMA(0, 0, At, B0); BAR; SCHED;
            LDB(B1, 0, 1); STAGE(SB(0, 0), b2, voffB);
            BAR; WAIT_L(0); MMA(0, 1, At, B1); BAR;
            LDA(At, 0, 1); STAGE(SA(0, 0), a2, voffA);
            BAR; WAIT_L(0); MMA(1, 0, At, B0); BAR; SCHED;
            STAGE(SB(0, 1), b2 + hstep, voffB);
            WAIT_V(6); BAR; MMA(1, 1, At, B1); BAR;
            LDB(B0, 1, 0); SCHED; LDA(At, 1, 0); STAGE(SA(0, 1), a2 + hstep, voffA);
            WAIT_L(8); BAR; WAIT_L(0); MMA(0, 0, At, B0); BAR; SCHED;
            LDB(B1, 1, 1); STAGE(SB(1, 0), b3, voffB);
            BAR; WAIT_L(0); MMA(0, 1, At, B1); BAR;
            LDA(At, 1, 1); STAGE(SA(1, 0), a3, voffA);
            BAR; WAIT_L(0); MMA(1, 0, At, B0); BAR; SCHED;
            STAGE(SB(1, 1), b3 + hstep, voffB);
            WAIT_V(6); BAR; MMA(1, 1, At, B1); BAR;
        }
        STAMP(myws, 3 + 2 * h);
        if (h == 1) {
            const int rt = (pm * 2 + (pn1 - 14)) * (RS - 1);
            if (part) {
                const __amdgpu_buffer_rsrc_t rs = mk_rsrc(myws + OFF_P + (size_t)(rt + part - 1) * 262144, 262144);
#pragma unroll
                for (int i = 0; i < 32; ++i)
                    __builtin_amdgcn_raw_buffer_store_b128(__builtin_bit_cast(u32x4, acc[i >> 4][(i >> 3) & 1][(i >> 1) & 3][i & 1]), rs, tid * 16, i * 8192, 16);
                asm volatile("s_waitcnt vmcnt(0)" ::: "memory");
                if (lane == 0) __hip_atomic_fetch_add(ctl(myws, C_PART + rt + part - 1), 1u, __ATOMIC_RELAXED, __HIP_MEMORY_SCOPE_AGENT);
                break;
            }
            for (int pt = 0; pt < RS - 1; ++pt) {
                if (lane == 0) spin_ge<4>(ctl(myws, C_PART + rt + pt), 8u * call, tmo, call);
                asm volatile("" ::: "memory");
                const __amdgpu_buffer_rsrc_t rs = mk_rsrc(myws + OFF_P + (size_t)(rt + pt) * 262144, 262144);
#pragma unroll
                for (int i = 0; i < 32; ++i) { acc[i >> 4][(i >> 3) & 1][(i >> 1) & 3][i & 1] += __builtin_bit_cast(f32x4, __builtin_amdgcn_raw_buffer_load_b128(rs, tid * 16, i * 8192, 0));
                    if ((i & 7) == 7) asm volatile("" ::: "memory"); }
            }
        }
        {
            const bool own = (pm >> 1) == rank;
            const int row0 = wr * 64 + fr, col0 = wc * 32 + 8 * fq;
            if (own) {
                bf16_t* base = p.out + (size_t)((pm & 1) * 256) * NTOT + (size_t)rank * NS + pn * 256;
#pragma unroll
                for (int ai = 0; ai < 2; ++ai)
#pragma unroll
                    for (int m = 0; m < 4; ++m) { bf16_t* rowp = base + (size_t)(row0 + ai * HALF + m * 16) * NTOT + col0;
#pragma unroll
                        for (int bj = 0; bj < 2; ++bj) *(u32x4*)(rowp + bj * HALF) = pack8(acc[ai][bj][m][0], acc[ai][bj][m][1]); }
            } else {
                const __amdgpu_buffer_rsrc_t rs = mk_rsrc(myws + OFF_O + ((size_t)(pm * 256) * NS + pn * 256) * 2, 256 * NS * 2);
#pragma unroll
                for (int ai = 0; ai < 2; ++ai)
#pragma unroll
                    for (int m = 0; m < 4; ++m) { const int ro = ((row0 + ai * HALF + m * 16) * NS + col0) * 2;
#pragma unroll
                        for (int bj = 0; bj < 2; ++bj) __builtin_amdgcn_raw_buffer_store_b128(pack8(acc[ai][bj][m][0], acc[ai][bj][m][1]), rs, ro + bj * HALF * 2, 0, 16); }
                asm volatile("s_waitcnt vmcnt(0)" ::: "memory");
                if (lane == 0) __hip_atomic_fetch_add(ctl(myws, C_TILE + pm * 16 + pn), 1u, __ATOMIC_RELAXED, __HIP_MEMORY_SCOPE_AGENT);
            }
        }
        STAMP(myws, 4 + 2 * h);
#pragma unroll
        for (int a = 0; a < 2; ++a)
#pragma unroll
            for (int b = 0; b < 2; ++b)
#pragma unroll
                for (int m = 0; m < 4; ++m)
#pragma unroll
                    for (int n = 0; n < 2; ++n) acc[a][b][m][n] = (f32x4){0.f, 0.f, 0.f, 0.f};
    }
    WAIT_V(0);
    if (wr == 0) BAR;
    BAR;
#undef SA
#undef SB
#undef STAGE
#undef LDA
#undef LDB
#undef MMA
#undef WAIT_V
#undef WAIT_L
#undef BAR
#undef SCHED
}

__global__ __launch_bounds__(512, 2) void fused_kernel(Params p) {
    extern __shared__ __attribute__((aligned(16))) unsigned char shm[];
    LAS unsigned char* lds = (LAS unsigned char*)shm;
    LAS unsigned* lw = (LAS unsigned*)(lds + STAGE_BYTES);
    const int grp = (int)blockIdx.x & 7, slot = (int)blockIdx.x >> 3;
    STAMP(p.ws_all[p.rank], 0);
    if (threadIdx.x == 0) {
        const unsigned t = __hip_atomic_fetch_add(ctl(p.ws_all[p.rank], C_TICKET), 1u, __ATOMIC_RELAXED, __HIP_MEMORY_SCOPE_AGENT);
        lw[0] = t / (unsigned)NBLK + 1u;
        __builtin_amdgcn_fence(__ATOMIC_ACQUIRE, "");
        asm volatile("s_waitcnt vmcnt(0)" ::: "memory");
    }
    __syncthreads();
    const unsigned call = __builtin_amdgcn_readfirstlane(lw[0]);
    __syncthreads();
    if (slot < NC) comm_role(p, lds, call, (p.rank + grp) & 7, slot);
    else gemm_role(p, lds, call, grp, slot - NC);
}

}

extern "C" void kernel_launch(int rank, int nranks, ncclComm_t comm,
                              hipStream_t stream,
                              void* const* d_in, const int* in_sizes, int n_in,
                              void* d_out, int out_size,
                              const int* in_dtypes, int out_dtype,
                              void* const* d_out_all,
                              void* const* d_ws_all, size_t ws_bytes) {
    if (nranks != NR || ws_bytes < WS_NEED || (unsigned)rank >= (unsigned)NR) { fprintf(stderr, "unexpected configuration\n"); return; }
    static bool attr[64] = {};
    int dev = 0; HIP_CHECK(hipGetDevice(&dev));
    if (!attr[dev & 63]) { HIP_CHECK(hipFuncSetAttribute((const void*)fused_kernel, hipFuncAttributeMaxDynamicSharedMemorySize, LDS_BYTES)); attr[dev & 63] = true; }
    Params p;
    p.x = (const bf16_t*)d_in[0]; p.w = (const bf16_t*)d_in[1]; p.out = (bf16_t*)d_out;
    p.out_all = d_out_all; p.ws_all = d_ws_all; p.rank = rank;
    hipLaunchKernelGGL(fused_kernel, dim3(NBLK), dim3(512), LDS_BYTES, stream, p);
    HIP_CHECK(hipGetLastError());
}
```
